```python
import numpy as np

kernel_name = "gpu_gemm_rs_m4096_k4096_n32768_bf16_relu"


ABSMAX_RTOL = 0.008

COMPUTE_DTYPE = "bfloat16"
EPILOGUE = "relu"
IN_QUANT = None
OUT_QUANT = None
IN_DTYPES = {"x": "bfloat16", "w_mat": "bfloat16"}
WORLD_SIZE = 8

def _to_bf16(x):
    u = np.ascontiguousarray(x, dtype=np.float32).view(np.uint32)
    return ((u + 0x7FFF + ((u >> 16) & 1)) & 0xFFFF0000).view(np.float32)

def setup_inputs():

    x = _to_bf16(np.random.randn(4096, 4096).astype(np.float32))
    w_mat = _to_bf16((np.random.randn(4096, 32768) / np.sqrt(4096)).astype(np.float32))
    return {"x": x, "w_mat": w_mat}

def _epilogue(y):
    return np.maximum(y, 0.0).astype(np.float32)

def reference(x, w_mat):
    y = (x @ w_mat).astype(np.float32)
    return _epilogue(y)
```

```cpp
#include <hip/hip_runtime.h>
#include <rccl/rccl.h>
#include <cstdio>

#define HIP_CHECK(expr) do { hipError_t _e = (expr); if (_e != hipSuccess) { \
    fprintf(stderr, "HIP error %s at %s:%d\n", hipGetErrorString(_e), __FILE__, __LINE__); \
    return; } } while (0)

namespace {

typedef unsigned short bf16_t;
typedef short bf16x8 __attribute__((ext_vector_type(8)));
typedef float f32x4 __attribute__((ext_vector_type(4)));
typedef unsigned u32x4 __attribute__((ext_vector_type(4)));
typedef unsigned u32x2 __attribute__((ext_vector_type(2)));
#define LAS __attribute__((address_space(3)))

constexpr int NR = 8;
constexpr int GM = 4096;
constexpr int KC = 512;
constexpr int GN = 32768;
constexpr int NS = 4096;
constexpr int MS = 512;

constexpr size_t CHUNK_BYTES = (size_t)GM * KC * 2;
constexpr size_t XG_OFF = 0;
constexpr size_t WT_OFF = XG_OFF + NR * CHUNK_BYTES;
constexpr size_t FL_OFF = WT_OFF + NR * CHUNK_BYTES;
constexpr size_t IN_FLAG = FL_OFF;
constexpr size_t READY_FLAG = FL_OFF + 2048;
constexpr size_t OUT_FLAG = FL_OFF + 4096;
constexpr size_t TICKET0 = FL_OFF + 4096 + 256;
constexpr size_t TICKET1 = FL_OFF + 4096 + 512;
constexpr size_t TICKET2 = FL_OFF + 4096 + 768;
constexpr size_t TMO_OFF = FL_OFF + 4096 + 1024;
constexpr size_t WS_NEEDED = FL_OFF + 8192;
constexpr unsigned long long DEADLINE_TICKS = 100000000ull;

constexpr int BM = 256, BK = 64, HALF = 128, HTB = HALF * BK * 2, STAGE_BYTES = 8 * HTB, NXCD = 8, WGM = 8;
constexpr int NWG = 256;

struct Args {
    const bf16_t* x; const bf16_t* w;
    char* ws[NR]; char* out[NR];
    int rank;
};

__device__ __forceinline__ int lds_byte(int r, int c) { const int st = (r >> 4) * 2 + (c >> 5), rr = r & 15, cc = c & 31, ob = rr * 64 + cc * 2; return st * 1024 + (ob ^ (((ob >> 9) & 1) << 5)); }
__device__ __forceinline__ void stage_rc(int b, int& R, int& C) { const int st = b / 1024, sb = b % 1024, swz = sb ^ (((sb >> 9) & 1) << 5); R = (st >> 1) * 16 + swz / 64; C = (st & 1) * 32 + (swz % 64) / 2; }
__device__ __forceinline__ int perm32(int rho) { const int n = rho >> 4, i = rho & 15; return 8 * (i >> 2) + 4 * n + (i & 3); }
__device__ __forceinline__ unsigned cvt_pk_bf16(float lo, float hi) { unsigned r; asm volatile("v_cvt_pk_bf16_f32 %0, %1, %2" : "=v"(r) : "v"(lo), "v"(hi)); return r; }

__device__ __forceinline__ void wait_flag(const unsigned* flag, unsigned target, unsigned* tmo, unsigned call) {
    const unsigned long long t0 = __builtin_amdgcn_s_memrealtime();
    while (__hip_atomic_load(flag, __ATOMIC_RELAXED, __HIP_MEMORY_SCOPE_SYSTEM) < target) {
        if (__hip_atomic_load(tmo, __ATOMIC_RELAXED, __HIP_MEMORY_SCOPE_AGENT) == call ||
            __builtin_amdgcn_s_memrealtime() - t0 > DEADLINE_TICKS) {
            __hip_atomic_store(tmo, call, __ATOMIC_RELAXED, __HIP_MEMORY_SCOPE_AGENT);
            break;
        }
        __builtin_amdgcn_s_sleep(2);
    }
}

__device__ __forceinline__ unsigned draw_call(char* myws, size_t ticket_off, unsigned* lds_word) {
    if (threadIdx.x == 0) {
        const unsigned t = __hip_atomic_fetch_add((unsigned*)(myws + ticket_off), 1u, __ATOMIC_RELAXED, __HIP_MEMORY_SCOPE_AGENT);
        *lds_word = t / gridDim.x + 1u;
    }
    __syncthreads();
    const unsigned c = *lds_word;
    __syncthreads();
    return c;
}

__device__ void push_phase(const Args& a, unsigned char* lds) {
    const int tid = threadIdx.x, bid = blockIdx.x, rank = a.rank;
    char* myws = a.ws[rank];
    unsigned* tmo = (unsigned*)(myws + TMO_OFF);
    const unsigned call = draw_call(myws, TICKET0, (unsigned*)lds);
    if (tid < NR && tid != rank) wait_flag((const unsigned*)(myws + READY_FLAG + tid * 128), call - 1u, tmo, call);
    __syncthreads();
    {
        const u32x4* src = (const u32x4*)((const char*)a.x + (size_t)bid * 16384);
        const u32x4 v0 = src[tid], v1 = src[tid + 512];
#pragma unroll
        for (int d = 1; d < NR; ++d) {
            const int p = (rank + d) & 7;
            u32x4* dst = (u32x4*)(a.ws[p] + XG_OFF + (size_t)rank * CHUNK_BYTES + (size_t)bid * 16384);
            dst[tid] = v0; dst[tid + 512] = v1;
        }
    }
    constexpr int RS = 264;
    const int ng = tid & 15, kq = tid >> 4;
    for (int it = 0; it < 4; ++it) {
        const int tile = it * NWG + bid;
        const int d = tile & 7, rest = tile >> 3;
        const int p = (rank + d) & 7, ntile = rest & 31, ktile = rest >> 5;
        const bf16_t* src = a.w + (size_t)(ktile * 128 + kq * 4) * GN + (size_t)p * NS + ntile * 128 + ng * 8;
        u32x4 r0 = *(const u32x4*)(src), r1 = *(const u32x4*)(src + GN), r2 = *(const u32x4*)(src + 2 * GN), r3 = *(const u32x4*)(src + 3 * GN);
        if (it) __syncthreads();
#pragma unroll
        for (int j = 0; j < 4; ++j) {
            u32x2 lo, hi;
            lo.x = __builtin_amdgcn_perm(r1[j], r0[j], 0x05040100u); lo.y = __builtin_amdgcn_perm(r3[j], r2[j], 0x05040100u);
            hi.x = __builtin_amdgcn_perm(r1[j], r0[j], 0x07060302u); hi.y = __builtin_amdgcn_perm(r3[j], r2[j], 0x07060302u);
            *(u32x2*)(lds + (ng * 8 + 2 * j) * RS + kq * 8) = lo;
            *(u32x2*)(lds + (ng * 8 + 2 * j + 1) * RS + kq * 8) = hi;
        }
        __syncthreads();
        char* dstb = a.ws[p] + WT_OFF + (size_t)rank * CHUNK_BYTES + (size_t)(ntile * 128) * (KC * 2) + ktile * 256;
#pragma unroll
        for (int i = 0; i < 4; ++i) {
            const int idx = i * 512 + tid, n = idx >> 4, c = idx & 15;
            const u32x2 q0 = *(const u32x2*)(lds + n * RS + c * 16), q1 = *(const u32x2*)(lds + n * RS + c * 16 + 8);
            u32x4 v; v.x = q0.x; v.y = q0.y; v.z = q1.x; v.w = q1.y;
            *(u32x4*)(dstb + (size_t)n * (KC * 2) + c * 16) = v;
        }
    }
    asm volatile("s_waitcnt vmcnt(0)" ::: "memory");
    __syncthreads();
    if (tid == 0) {
        __builtin_amdgcn_fence(__ATOMIC_RELEASE, "");
        asm volatile("s_waitcnt vmcnt(0)" ::: "memory");
#pragma unroll
        for (int d = 1; d < NR; ++d) {
            const int p = (rank + d) & 7;
            __hip_atomic_fetch_add((unsigned*)(a.ws[p] + IN_FLAG + rank * 128), 1u, __ATOMIC_RELAXED, __HIP_MEMORY_SCOPE_SYSTEM);
        }
    }
}

__device__ void gemm_phase(const Args& a, LAS unsigned char* lds, unsigned char* lds_generic) {
    const int tid = threadIdx.x, wid = __builtin_amdgcn_readfirstlane(tid >> 6), lane = tid & 63, wr = wid >> 2, wc = wid & 3, fr = lane & 15, fq = lane >> 4;
    const int rank = a.rank;
    char* myws = a.ws[rank];
    unsigned* tmo = (unsigned*)(myws + TMO_OFF);
    const unsigned call = draw_call(myws, TICKET1, (unsigned*)lds_generic);
    constexpr int K = KC, nt = K / BK;
    int pm, pn;
    { const int nM = GM / BM, nN = NS / BM, nwg = nM * nN; int wgid = blockIdx.x;
      const int q = nwg / NXCD, xcd = wgid % NXCD, off = wgid / NXCD; wgid = xcd * q + off;
      const int nig = WGM * nN, gid = wgid / nig, fm = gid * WGM; pm = fm + ((wgid % nig) % WGM); pn = (wgid % nig) / WGM; }
    unsigned voffA[2], voffB[2];
#pragma unroll
    for (int i = 0; i < 2; ++i) { int R, C; stage_rc(tid * 16 + i * 8192, R, C); const int Rb = (R & ~31) + perm32(R & 31);
        voffA[i] = (unsigned)(R * K + C) * 2u; voffB[i] = (unsigned)(Rb * K + C) * 2u; }
    const size_t kstep = (size_t)(BK * 2), hstep = (size_t)HALF * K * 2, tstep = 2 * hstep;
    const unsigned ldsw = (unsigned)wid * 1024u;
    const int aoff = lds_byte(wr * 64 + fr, fq * 8), boff = lds_byte(wc * 32 + fr, fq * 8);
#define SA(b, h) (((b) * 2 + (h)) * HTB)
#define SB(b, h) ((4 + (b) * 2 + (h)) * HTB)
#define STAGE(bufoff, gbase, voff) do { _Pragma("unroll") for (int _i = 0; _i < 2; ++_i) \
        __builtin_amdgcn_global_load_lds((const unsigned*)((const char*)(gbase) + (voff)[_i]), (LAS unsigned*)(lds + (bufoff) + ldsw + _i * 8192), 16, 0, 0); } while (0)
#define LDA(dst, b, h) do { _Pragma("unroll") for (int m = 0; m < 4; ++m) _Pragma("unroll") for (int k = 0; k < 2; ++k) dst[m][k] = *(const LAS bf16x8*)(lds + SA(b, h) + aoff + m * 2048 + k * 1024); } while (0)
#define LDB(dst, b, h) do { _Pragma("unroll") for (int n = 0; n < 2; ++n) _Pragma("unroll") for (int k = 0; k < 2; ++k) dst[n][k] = *(const LAS bf16x8*)(lds + SB(b, h) + boff + n * 2048 + k * 1024); } while (0)
#define MMA(ai, bj, At, Bt) do { __builtin_amdgcn_s_setprio(1); _Pragma("unroll") for (int m = 0; m < 4; ++m) _Pragma("unroll") for (int n = 0; n < 2; ++n) _Pragma("unroll") for (int k = 0; k < 2; ++k) \
        acc[ai][bj][m][n] = __builtin_amdgcn_mfma_f32_16x16x32_bf16(Bt[n][k], At[m][k], acc[ai][bj][m][n], 0, 0, 0); __builtin_amdgcn_s_setprio(0); } while (0)
#define WAIT_V(n) asm volatile("s_waitcnt vmcnt(" #n ")" ::: "memory")
#define WAIT_L(n) asm volatile("s_waitcnt lgkmcnt(" #n ")" ::: "memory")
#define BAR __builtin_amdgcn_s_barrier()
#define SCHED __builtin_amdgcn_sched_barrier(0)
    f32x4 acc[2][2][4][2];
#pragma unroll
    for (int i = 0; i < 2; ++i)
#pragma unroll
        for (int j = 0; j < 2; ++j)
#pragma unroll
            for (int m = 0; m < 4; ++m)
#pragma unroll
                for (int n = 0; n < 2; ++n) acc[i][j][m][n] = (f32x4){0.f, 0.f, 0.f, 0.f};
    bf16x8 At[4][2], B0[2][2], B1[2][2];
    const char* cA = (const char*)a.x + (size_t)pm * tstep;
    const char* cB = myws + WT_OFF + (size_t)rank * CHUNK_BYTES + (size_t)pn * tstep;
    STAGE(SB(0, 0), cB, voffB); STAGE(SA(0, 0), cA, voffA); STAGE(SB(0, 1), cB + hstep, voffB); STAGE(SA(0, 1), cA + hstep, voffA);
    if (wr == 1) BAR;
    WAIT_V(4); BAR;
    STAGE(SB(1, 0), cB + kstep, voffB); STAGE(SA(1, 0), cA + kstep, voffA); STAGE(SB(1, 1), cB + hstep + kstep, voffB);
    WAIT_V(6); BAR;
    for (int ui = 0; ui < NR; ++ui) {
        const bool has_next = ui + 1 < NR;
        const int nc = (rank + ui + 1) & 7;
        const char* nA = has_next ? myws + XG_OFF + (size_t)nc * CHUNK_BYTES + (size_t)pm * tstep : cA;
        const char* nB = has_next ? myws + WT_OFF + (size_t)nc * CHUNK_BYTES + (size_t)pn * tstep : cB;
        for (int t = 0; t < nt; t += 2) {
            const bool last = (t == nt - 2);
            const char* a1 = cA + (size_t)(t + 1) * kstep;
            const char* a2 = last ? nA : cA + (size_t)(t + 2) * kstep; const char* b2 = last ? nB : cB + (size_t)(t + 2) * kstep;
            const char* a3 = a2 + kstep; const char* b3 = b2 + kstep;
            if (last && has_next) {
                if (tid == 0) {
                    wait_flag((const unsigned*)(myws + IN_FLAG + nc * 128), call * (unsigned)NWG, tmo, call);
                    __builtin_amdgcn_fence(__ATOMIC_ACQUIRE, "");
                    asm volatile("s_waitcnt vmcnt(0)" ::: "memory");
                }
                asm volatile("" ::: "memory"); BAR; asm volatile("" ::: "memory");
            }
            LDB(B0, 0, 0); SCHED; LDA(At, 0, 0); STAGE(SA(1, 1), a1 + hstep, voffA);
            WAIT_L(8); BAR; WAIT_L(0); MMA(0, 0, At, B0); BAR; SCHED;
            LDB(B1, 0, 1); STAGE(SB(0, 0), b2, voffB);
            BAR; WAIT_L(0); MMA(0, 1, At, B1); BAR;
            LDA(At, 0, 1); STAGE(SA(0, 0), a2, voffA);
            BAR; WAIT_L(0); MMA(1, 0, At, B0); BAR; SCHED;
            STAGE(SB(0, 1), b2 + hstep, voffB);
            WAIT_V(6); BAR; MMA(1, 1, At, B1); BAR;
            LDB(B0, 1, 0); SCHED; LDA(At, 1, 0); STAGE(SA(0, 1), a2 + hstep, voffA);
            WAIT_L(8); BAR; WAIT_L(0); MMA(0, 0, At, B0); BAR; SCHED;
            LDB(B1, 1, 1); STAGE(SB(1, 0), b3, voffB);
            BAR; WAIT_L(0); MMA(0, 1, At, B1); BAR;
            LDA(At, 1, 1); STAGE(SA(1, 0), a3, voffA);
            BAR; WAIT_L(0); MMA(1, 0, At, B0); BAR; SCHED;
            STAGE(SB(1, 1), b3 + hstep, voffB);
            WAIT_V(6); BAR; MMA(1, 1, At, B1); BAR;
        }
        cA = nA; cB = nB;
    }
    WAIT_V(0);
    if (wr == 0) BAR;
    BAR;
    const bool bad = __hip_atomic_load(tmo, __ATOMIC_RELAXED, __HIP_MEMORY_SCOPE_AGENT) == call;
    const int owner = pm >> 1;
    {
        bf16_t* obase = (bf16_t*)a.out[owner] + (size_t)((pm & 1) * BM + wr * 64 + fr) * GN + (size_t)rank * NS + pn * BM + wc * 32 + 8 * fq;
#pragma unroll
        for (int ai = 0; ai < 2; ++ai)
#pragma unroll
            for (int m = 0; m < 4; ++m) {
                bf16_t* rowp = obase + (size_t)(ai * HALF + m * 16) * GN;
#pragma unroll
                for (int bj = 0; bj < 2; ++bj) {
                    const f32x4 v0 = acc[ai][bj][m][0], v1 = acc[ai][bj][m][1];
                    u32x4 w;
                    w.x = cvt_pk_bf16(fmaxf(v0[0], 0.f), fmaxf(v0[1], 0.f)); w.y = cvt_pk_bf16(fmaxf(v0[2], 0.f), fmaxf(v0[3], 0.f));
                    w.z = cvt_pk_bf16(fmaxf(v1[0], 0.f), fmaxf(v1[1], 0.f)); w.w = cvt_pk_bf16(fmaxf(v1[2], 0.f), fmaxf(v1[3], 0.f));
                    if (bad) { w.x = 0x7fc07fc0u; w.y = 0x7fc07fc0u; w.z = 0x7fc07fc0u; w.w = 0x7fc07fc0u; }
                    *(u32x4*)(rowp + bj * HALF) = w;
                }
            }
    }
    asm volatile("s_waitcnt vmcnt(0)" ::: "memory");
    __syncthreads();
    if (tid == 0 && owner != rank) {
        __builtin_amdgcn_fence(__ATOMIC_RELEASE, "");
        asm volatile("s_waitcnt vmcnt(0)" ::: "memory");
        __hip_atomic_fetch_add((unsigned*)(a.ws[owner] + OUT_FLAG), 1u, __ATOMIC_RELAXED, __HIP_MEMORY_SCOPE_SYSTEM);
    }
#undef SA
#undef SB
#undef STAGE
#undef LDA
#undef LDB
#undef MMA
}

__device__ void finish_phase(const Args& a, unsigned char* lds) {
    const int tid = threadIdx.x, rank = a.rank;
    char* myws = a.ws[rank];
    unsigned* tmo = (unsigned*)(myws + TMO_OFF);
    const unsigned call = draw_call(myws, TICKET2, (unsigned*)lds);
    if (tid < NR && tid != rank)
        __hip_atomic_store((unsigned*)(a.ws[tid] + READY_FLAG + rank * 128), call, __ATOMIC_RELAXED, __HIP_MEMORY_SCOPE_SYSTEM);
    if (tid == 0) {
        wait_flag((const unsigned*)(myws + OUT_FLAG), call * (unsigned)((NR - 1) * 32), tmo, call);
        __builtin_amdgcn_fence(__ATOMIC_ACQUIRE, "");
        if (__hip_atomic_load(tmo, __ATOMIC_RELAXED, __HIP_MEMORY_SCOPE_AGENT) == call)
            ((bf16_t*)a.out[rank])[(size_t)rank * NS] = 0x7fc0;
    }
}

__global__ __launch_bounds__(512, 2) void fused_kernel(Args a, int phase) {
    __shared__ __attribute__((aligned(16))) unsigned char lds[STAGE_BYTES];
    if (phase == 0) push_phase(a, lds);
    else if (phase == 1) gemm_phase(a, (LAS unsigned char*)lds, lds);
    else finish_phase(a, lds);
}

}

extern "C" void kernel_launch(int rank, int nranks, ncclComm_t comm,
                              hipStream_t stream,
                              void* const* d_in, const int* in_sizes, int n_in,
                              void* d_out, int out_size,
                              const int* in_dtypes, int out_dtype,
                              void* const* d_out_all,
                              void* const* d_ws_all, size_t ws_bytes) {
    if (nranks != NR || ws_bytes < WS_NEEDED || (unsigned)rank >= (unsigned)NR) { fprintf(stderr, "unexpected configuration\n"); return; }
    Args a;
    a.x = (const bf16_t*)d_in[0]; a.w = (const bf16_t*)d_in[1]; a.rank = rank;
    for (int p = 0; p < NR; ++p) { a.ws[p] = (char*)d_ws_all[p]; a.out[p] = (char*)d_out_all[p]; }
    fused_kernel<<<NWG, 512, 0, stream>>>(a, 0);
    fused_kernel<<<NWG, 512, 0, stream>>>(a, 1);
    fused_kernel<<<1, 512, 0, stream>>>(a, 2);
    HIP_CHECK(hipGetLastError());
}
```

```cpp
#include <hip/hip_runtime.h>
#include <rccl/rccl.h>
#include <cstdio>

#define HIP_CHECK(expr) do { hipError_t _e = (expr); if (_e != hipSuccess) { \
    fprintf(stderr, "HIP error %s at %s:%d\n", hipGetErrorString(_e), __FILE__, __LINE__); \
    return; } } while (0)

namespace {

typedef unsigned short bf16_t;
typedef short bf16x8 __attribute__((ext_vector_type(8)));
typedef float f32x4 __attribute__((ext_vector_type(4)));
typedef unsigned u32x4 __attribute__((ext_vector_type(4)));
typedef unsigned u32x2 __attribute__((ext_vector_type(2)));
#define LAS __attribute__((address_space(3)))
typedef short v4i16_t __attribute__((ext_vector_type(4)));

constexpr int NR = 8;
constexpr int GM = 4096;
constexpr int KC = 512;
constexpr int GN = 32768;
constexpr int NS = 4096;
constexpr int MS = 512;

constexpr size_t CHUNK_BYTES = (size_t)GM * KC * 2;
constexpr size_t XG_OFF = 0;
constexpr size_t WT_OFF = XG_OFF + NR * CHUNK_BYTES;
constexpr size_t FL_OFF = WT_OFF + NR * CHUNK_BYTES;
constexpr size_t IN_FLAG = FL_OFF;
constexpr size_t READY_FLAG = FL_OFF + 4096;
constexpr size_t OUT_FLAG = FL_OFF + 8192;
constexpr size_t TICKET0 = FL_OFF + 8192 + 256;
constexpr size_t TICKET1 = FL_OFF + 8192 + 512;
constexpr size_t TICKET2 = FL_OFF + 8192 + 768;
constexpr size_t TMO_OFF = FL_OFF + 8192 + 1024;
constexpr size_t WS_NEEDED = FL_OFF + 16384;
constexpr unsigned long long DEADLINE_TICKS = 100000000ull;

constexpr int BM = 256, BK = 64, HALF = 128, HTB = HALF * BK * 2, STAGE_BYTES = 8 * HTB, NXCD = 8, WGM = 8;
constexpr int NWG = 256;

#define STAMP(ph, i) do {} while (0)
struct Args {
    const bf16_t* x; const bf16_t* w;
    char* const* ws; char* const* out;
    int rank;
};

__device__ __forceinline__ int lds_byte(int r, int c) { const int st = (r >> 4) * 2 + (c >> 5), rr = r & 15, cc = c & 31, ob = rr * 64 + cc * 2; return st * 1024 + (ob ^ (((ob >> 9) & 1) << 5)); }
__device__ __forceinline__ void stage_rc(int b, int& R, int& C) { const int st = b / 1024, sb = b % 1024, swz = sb ^ (((sb >> 9) & 1) << 5); R = (st >> 1) * 16 + swz / 64; C = (st & 1) * 32 + (swz % 64) / 2; }
__device__ __forceinline__ int perm32(int rho) { const int n = rho >> 4, i = rho & 15; return 8 * (i >> 2) + 4 * n + (i & 3); }
__device__ __forceinline__ unsigned cvt_pk_bf16(float lo, float hi) { unsigned r; asm volatile("v_cvt_pk_bf16_f32 %0, %1, %2" : "=v"(r) : "v"(lo), "v"(hi)); return r; }

__device__ __forceinline__ unsigned long long uni64(const void* p) {
    const unsigned long long v = (unsigned long long)p;
    const unsigned lo = __builtin_amdgcn_readfirstlane((unsigned)v), hi = __builtin_amdgcn_readfirstlane((unsigned)(v >> 32));
    return ((unsigned long long)hi << 32) | lo;
}
__device__ __forceinline__ void store_wt(char* p, u32x4 v) { asm volatile("global_store_dwordx4 %0, %1, off sc0 sc1\n\ts_nop 1" :: "v"(p), "v"(v) : "memory"); }
__device__ __forceinline__ void wait_flag(const unsigned* flag, unsigned target, unsigned* tmo, unsigned call) {
    const unsigned long long t0 = __builtin_amdgcn_s_memrealtime();
    while (__hip_atomic_load(flag, __ATOMIC_RELAXED, __HIP_MEMORY_SCOPE_SYSTEM) < target) {
        if (__hip_atomic_load(tmo, __ATOMIC_RELAXED, __HIP_MEMORY_SCOPE_AGENT) == call ||
            __builtin_amdgcn_s_memrealtime() - t0 > DEADLINE_TICKS) {
            __hip_atomic_store(tmo, call, __ATOMIC_RELAXED, __HIP_MEMORY_SCOPE_AGENT);
            break;
        }
        __builtin_amdgcn_s_sleep(2);
    }
}

__device__ __forceinline__ unsigned draw_call(char* myws, size_t ticket_off, unsigned* lds_word) {
    if (threadIdx.x == 0) {
        const unsigned t = __hip_atomic_fetch_add((unsigned*)(myws + ticket_off), 1u, __ATOMIC_RELAXED, __HIP_MEMORY_SCOPE_AGENT);
        *lds_word = t / gridDim.x + 1u;
    }
    __syncthreads();
    const unsigned c = *lds_word;
    __syncthreads();
    return c;
}

__device__ void gemm_phase(const Args& a, LAS unsigned char* lds, unsigned char* lds_generic) {
    const int tid = threadIdx.x, wid = __builtin_amdgcn_readfirstlane(tid >> 6), lane = tid & 63, wr = wid >> 2, wc = wid & 3, fr = lane & 15, fq = lane >> 4;
    const int rank = a.rank;
    char* myws = a.ws[rank];
    unsigned* tmo = (unsigned*)(myws + TMO_OFF);
    STAMP(1, 0);
    const unsigned call = draw_call(myws, TICKET1, (unsigned*)lds_generic);
    constexpr int K = KC;
    const int bid = blockIdx.x;
    if (tid < NR && tid != rank) wait_flag((const unsigned*)(myws + READY_FLAG + tid * 128), call - 1u, tmo, call);
    __syncthreads();
    const unsigned vx = (unsigned)(lane >> 4) * 1024u + (unsigned)(lane & 15) * 16u, vw = (unsigned)lane * 16u;
    auto push_addr = [&](int pi, const char*& sp, char*& dp, unsigned& voff) {
        const int sub = pi / 7, g = 56 * bid + 8 * (pi % 7) + wid;
        const int p = __builtin_amdgcn_readfirstlane((rank + 1 + (g >> 11)) & 7), r = g & 2047;
        if (r < 1024) { const size_t o = (size_t)(4 * r) * 1024 + sub * 256;
            sp = (const char*)a.x + o; dp = a.ws[p] + XG_OFF + (size_t)rank * CHUNK_BYTES + o; voff = vx; }
        else { const int row = 128 * sub + ((r - 1024) >> 3); const int cb = (r & 7) * 1024;
            sp = (const char*)a.w + (size_t)row * (GN * 2) + (size_t)p * (NS * 2) + cb; dp = a.ws[p] + WT_OFF + (size_t)rank * CHUNK_BYTES + (size_t)row * (NS * 2) + cb; voff = vw; }
    };
    auto signal_sub = [&](int sub) {
        if (lane == 0) { __builtin_amdgcn_fence(__ATOMIC_RELEASE, ""); asm volatile("s_waitcnt vmcnt(0)" ::: "memory"); }
        const int d0 = (56 * bid) >> 11, d1 = (56 * bid + 55) >> 11;
        if (lane == 0 || (lane == 1 && d1 != d0)) { asm volatile("" ::: "memory");
            __hip_atomic_fetch_add((unsigned*)(a.ws[(rank + 1 + (lane ? d1 : d0)) & 7] + IN_FLAG + (rank * 4 + sub) * 128), 1u, __ATOMIC_RELAXED, __HIP_MEMORY_SCOPE_SYSTEM); }
    };
    {
        u32x4 v[7];
#pragma unroll
        for (int j = 0; j < 7; ++j) { const char* sp; char* dp; unsigned vo; push_addr(j, sp, dp, vo); v[j] = *(const u32x4*)(sp + vo); }
#pragma unroll
        for (int j = 0; j < 7; ++j) { const char* sp; char* dp; unsigned vo; push_addr(j, sp, dp, vo); store_wt(dp + vo, v[j]); }
        asm volatile("s_waitcnt vmcnt(0)" ::: "memory");
        __syncthreads();
        if (wid == 0) signal_sub(0);
    }
    u32x4 pA = {0u, 0u, 0u, 0u}, pB = {0u, 0u, 0u, 0u};
    unsigned long long arrived = 0ull;
    const unsigned* flagp = (const unsigned*)(myws + IN_FLAG + ((((rank + 1 + lane % 7) & 7) * 4 + (lane / 7 & 3)) * 128));
    const unsigned expect = call * (unsigned)((2048 * (6 - lane % 7) + 2047) / 56 - (2048 * (6 - lane % 7)) / 56 + 1);
    int pm, pn;
    { const int nM = GM / BM, nN = NS / BM, nwg = nM * nN; int wgid = blockIdx.x;
      const int q = nwg / NXCD, xcd = wgid % NXCD, off = wgid / NXCD; wgid = xcd * q + off;
      const int nig = WGM * nN, gid = wgid / nig, fm = gid * WGM; pm = fm + ((wgid % nig) % WGM); pn = (wgid % nig) / WGM; }
    unsigned voffA[2], krowB[2], colB[2];
#pragma unroll
    for (int i = 0; i < 2; ++i) { int R, C; stage_rc(tid * 16 + i * 8192, R, C); voffA[i] = (unsigned)(R * K + C) * 2u;
        const int k = wid * 4 + i * 32 + (lane >> 4), gi = lane & 15, f = (k & 3) | (((k >> 3) & 1) << 2);
        krowB[i] = (unsigned)k; colB[i] = (unsigned)((((gi >> 1) ^ f) << 1) | (gi & 1)) * 16u; }
    const size_t kstep = (size_t)(BK * 2), hstep = (size_t)HALF * K * 2, tstep = 2 * hstep;
    const unsigned ldsw = (unsigned)wid * 1024u;
    const int aoff = lds_byte(wr * 64 + fr, fq * 8);
    const int boff = (8 * fq + (fr >> 2)) * 256 + (((wc * 2 + ((fr & 3) >> 1)) ^ ((fr >> 2) | ((fq & 1) << 2))) * 32) + (fr & 1) * 16;
    const unsigned bbase = (unsigned)(size_t)(lds + 4 * HTB + boff);
#define SA(b, h) (((b) * 2 + (h)) * HTB)
#define SB(b, h) ((4 + (b) * 2 + (h)) * HTB)
#define STAGE(bufoff, gbase, voff) do { _Pragma("unroll") for (int _i = 0; _i < 2; ++_i) \
        __builtin_amdgcn_global_load_lds((const unsigned*)((const char*)(gbase) + (voff)[_i]), (LAS unsigned*)(lds + (bufoff) + ldsw + _i * 8192), 16, 0, 0); } while (0)
#define LDA(dst, b, h) do { _Pragma("unroll") for (int m = 0; m < 4; ++m) _Pragma("unroll") for (int k = 0; k < 2; ++k) dst[m][k] = *(const LAS bf16x8*)(lds + SA(b, h) + aoff + m * 2048 + k * 1024); } while (0)
#define TRRD(dst, off) asm volatile("ds_read_b64_tr_b16 %0, %1 offset:%2" : "=&v"(dst) : "v"(bbase), "i"(off) : "memory")
#define LDB(dst, b, h) do { _Pragma("unroll") for (int n = 0; n < 2; ++n) _Pragma("unroll") for (int k = 0; k < 2; ++k) { v4i16_t lo_, hi_; \
        TRRD(lo_, ((b) * 2 + (h)) * HTB + k * 8192 + n * 8); TRRD(hi_, ((b) * 2 + (h)) * HTB + k * 8192 + 1024 + n * 8); \
        dst[n][k] = __builtin_shufflevector(lo_, hi_, 0, 1, 2, 3, 4, 5, 6, 7); } } while (0)
#define MMA(ai, bj, At, Bt) do { __builtin_amdgcn_s_setprio(1); _Pragma("unroll") for (int m = 0; m < 4; ++m) _Pragma("unroll") for (int n = 0; n < 2; ++n) _Pragma("unroll") for (int k = 0; k < 2; ++k) \
        acc[ai][bj][m][n] = __builtin_amdgcn_mfma_f32_16x16x32_bf16(Bt[n][k], At[m][k], acc[ai][bj][m][n], 0, 0, 0); __builtin_amdgcn_s_setprio(0); } while (0)
#define WAIT_V(n) asm volatile("s_waitcnt vmcnt(" #n ")" ::: "memory")
#define WAIT_L(n) asm volatile("s_waitcnt lgkmcnt(" #n ")" ::: "memory")
#define BAR __builtin_amdgcn_s_barrier()
#define SCHED __builtin_amdgcn_sched_barrier(0)
#define PUSH_RATE 1
#define PUSH_S(j) ((j) >= PUSH_RATE && (j) < 21 + PUSH_RATE)
#define PUSH_L(j) ((j) < 21)
#define PUSH(reg, j) do { \
        if (PUSH_S(j)) { const char* sp_; char* dp_; unsigned vo_; push_addr((j) + 7 - PUSH_RATE, sp_, dp_, vo_); \
            asm volatile("s_nop 4\n\tglobal_store_dwordx4 %1, %0, %2 sc0 sc1\n\ts_nop 1" : "+v"(reg) : "v"(vo_), "s"(uni64(dp_)) : "memory"); } \
        if (PUSH_L(j)) { const char* sp_; char* dp_; unsigned vo_; push_addr((j) + 7, sp_, dp_, vo_); \
            asm volatile("s_nop 4\n\tglobal_load_dwordx4 %0, %1, %2" : "+v"(reg) : "v"(vo_), "s"(uni64(sp_)) : "memory"); } } while (0)
#define PUSH_WAIT(j) do { if (PUSH_S(j) && PUSH_L(j)) WAIT_V(8); else if (PUSH_S(j) || PUSH_L(j)) WAIT_V(7); else WAIT_V(6); } while (0)
    f32x4 acc[2][2][4][2];
#pragma unroll
    for (int i = 0; i < 2; ++i)
#pragma unroll
        for (int j = 0; j < 2; ++j)
#pragma unroll
            for (int m = 0; m < 4; ++m)
#pragma unroll
                for (int n = 0; n < 2; ++n) acc[i][j][m][n] = (f32x4){0.f, 0.f, 0.f, 0.f};
    bf16x8 At[4][2], B0[2][2], B1[2][2];
    constexpr int NU = 32;
    auto unit_ptrs = [&](int u, const char*& pa, const char*& pb, unsigned& strideB, int& src, int& sub) {
        if (u < 4) { src = rank; sub = u;
            pa = (const char*)a.x + (size_t)pm * tstep + sub * 256;
            pb = (const char*)a.w + (size_t)(sub * 128) * (GN * 2) + (size_t)(rank * NS + pn * BM) * 2; strideB = GN * 2; }
        else { const int v = u - 4; sub = v / 7; src = (rank + 1 + v % 7) & 7;
            pa = myws + XG_OFF + (size_t)src * CHUNK_BYTES + (size_t)pm * tstep + sub * 256;
            pb = myws + WT_OFF + (size_t)src * CHUNK_BYTES + (size_t)(sub * 128) * (NS * 2) + pn * (BM * 2); strideB = NS * 2; }
    };
    const char *cA, *nB; unsigned sB; int src_, sub_;
    unsigned voffB[2];
    unit_ptrs(0, cA, nB, sB, src_, sub_);
    voffB[0] = krowB[0] * sB + colB[0]; voffB[1] = krowB[1] * sB + colB[1];
    STAGE(SB(0, 0), nB, voffB); STAGE(SA(0, 0), cA, voffA); STAGE(SB(0, 1), nB + 256, voffB); STAGE(SA(0, 1), cA + hstep, voffA);
    if (wr == 1) BAR;
    WAIT_V(4); BAR;
    STAGE(SB(1, 0), nB + (size_t)64 * sB, voffB); STAGE(SA(1, 0), cA + kstep, voffA); STAGE(SB(1, 1), nB + (size_t)64 * sB + 256, voffB);
    WAIT_V(6); BAR;
    for (int u = 0; u < NU; ++u) {
        const bool has_next = u + 1 < NU;
        const char* nA;
        unit_ptrs(has_next ? u + 1 : u, nA, nB, sB, src_, sub_);
        voffB[0] = krowB[0] * sB + colB[0]; voffB[1] = krowB[1] * sB + colB[1];
        const char* a1 = cA + kstep;
        const char* a2 = nA; const char* b2 = nB;
        const char* a3 = a2 + kstep; const char* b3 = b2 + (size_t)64 * sB;
        if (wid == 0) {
            if (u == 9) signal_sub(1);
            if (u == 16) signal_sub(2);
            if (u == 23) signal_sub(3);
            if (u >= 3 && has_next && !((arrived >> (u - 3)) & 1ull)) {
                const unsigned long long t0 = __builtin_amdgcn_s_memrealtime();
                for (;;) {
                    const unsigned v = lane < 28 ? __hip_atomic_load(flagp, __ATOMIC_RELAXED, __HIP_MEMORY_SCOPE_SYSTEM) : 0u;
                    arrived = __builtin_amdgcn_ballot_w64(v >= expect);
                    if ((arrived >> (u - 3)) & 1ull) break;
                    if (__hip_atomic_load(tmo, __ATOMIC_RELAXED, __HIP_MEMORY_SCOPE_AGENT) == call || __builtin_amdgcn_s_memrealtime() - t0 > DEADLINE_TICKS) {
                        __hip_atomic_store(tmo, call, __ATOMIC_RELAXED, __HIP_MEMORY_SCOPE_AGENT); arrived = ~0ull; break; }
                    __builtin_amdgcn_s_sleep(1);
                }
                __builtin_amdgcn_fence(__ATOMIC_ACQUIRE, "");
                asm volatile("s_waitcnt vmcnt(0)" ::: "memory");
            }
        }
        asm volatile("" ::: "memory"); BAR; asm volatile("" ::: "memory");
        LDB(B0, 0, 0); SCHED; LDA(At, 0, 0); STAGE(SA(1, 1), a1 + hstep, voffA);
        WAIT_L(8); BAR; WAIT_L(0); SCHED; MMA(0, 0, At, B0); BAR; SCHED;
        LDB(B1, 0, 1); STAGE(SB(0, 0), b2, voffB);
        BAR; WAIT_L(0); SCHED; MMA(0, 1, At, B1); BAR;
        LDA(At, 0, 1); STAGE(SA(0, 0), a2, voffA);
        BAR; WAIT_L(0); SCHED; MMA(1, 0, At, B0); BAR; SCHED;
        STAGE(SB(0, 1), b2 + 256, voffB);
        WAIT_V(6);
        BAR; MMA(1, 1, At, B1); BAR;
        LDB(B0, 1, 0); SCHED; LDA(At, 1, 0); STAGE(SA(0, 1), a2 + hstep, voffA);
        PUSH(pA, u);
        WAIT_L(8); BAR; WAIT_L(0); SCHED; MMA(0, 0, At, B0); BAR; SCHED;
        LDB(B1, 1, 1); STAGE(SB(1, 0), b3, voffB);
        BAR; WAIT_L(0); SCHED; MMA(0, 1, At, B1); BAR;
        LDA(At, 1, 1); STAGE(SA(1, 0), a3, voffA);
        BAR; WAIT_L(0); SCHED; MMA(1, 0, At, B0); BAR; SCHED;
        STAGE(SB(1, 1), b3 + 256, voffB);
        PUSH_WAIT(u);
        BAR; MMA(1, 1, At, B1); BAR;
        cA = nA;
        if ((u & 3) == 3) STAMP(1, 2 + (u >> 2));
    }
    WAIT_V(0);
    if (wr == 0) BAR;
    BAR;
    const bool bad = __hip_atomic_load(tmo, __ATOMIC_RELAXED, __HIP_MEMORY_SCOPE_AGENT) == call;
    const int owner = pm >> 1;
    {
        bf16_t* obase = (bf16_t*)a.out[owner] + (size_t)((pm & 1) * BM + wr * 64 + fr) * GN + (size_t)rank * NS + pn * BM + wc * 32 + 8 * fq;
#pragma unroll
        for (int ai = 0; ai < 2; ++ai)
#pragma unroll
            for (int m = 0; m < 4; ++m) {
                bf16_t* rowp = obase + (size_t)(ai * HALF + m * 16) * GN;
#pragma unroll
                for (int bj = 0; bj < 2; ++bj) {
                    const f32x4 v0 = acc[ai][bj][m][0], v1 = acc[ai][bj][m][1];
                    u32x4 w;
                    w.x = cvt_pk_bf16(fmaxf(v0[0], 0.f), fmaxf(v0[1], 0.f)); w.y = cvt_pk_bf16(fmaxf(v0[2], 0.f), fmaxf(v0[3], 0.f));
                    w.z = cvt_pk_bf16(fmaxf(v1[0], 0.f), fmaxf(v1[1], 0.f)); w.w = cvt_pk_bf16(fmaxf(v1[2], 0.f), fmaxf(v1[3], 0.f));
                    if (bad) { w.x = 0x7fc07fc0u; w.y = 0x7fc07fc0u; w.z = 0x7fc07fc0u; w.w = 0x7fc07fc0u; }
                    store_wt((char*)(rowp + bj * HALF), w);
                }
            }
    }
    STAMP(1, 10);
    asm volatile("s_waitcnt vmcnt(0)" ::: "memory");
    __syncthreads();
    STAMP(1, 11);
    if (tid == 0 && owner != rank) {
        __builtin_amdgcn_fence(__ATOMIC_RELEASE, "");
        asm volatile("s_waitcnt vmcnt(0)" ::: "memory");
        __hip_atomic_fetch_add((unsigned*)(a.ws[owner] + OUT_FLAG), 1u, __ATOMIC_RELAXED, __HIP_MEMORY_SCOPE_SYSTEM);
    }
    STAMP(1, 12);
#undef SA
#undef SB
#undef STAGE
#undef LDA
#undef LDB
#undef MMA
}

__device__ void finish_phase(const Args& a, unsigned char* lds) {
    const int tid = threadIdx.x, rank = a.rank;
    char* myws = a.ws[rank];
    unsigned* tmo = (unsigned*)(myws + TMO_OFF);
    const unsigned call = draw_call(myws, TICKET2, (unsigned*)lds);
    if (tid < NR && tid != rank)
        __hip_atomic_store((unsigned*)(a.ws[tid] + READY_FLAG + rank * 128), call, __ATOMIC_RELAXED, __HIP_MEMORY_SCOPE_SYSTEM);
    if (tid == 0) {
        wait_flag((const unsigned*)(myws + OUT_FLAG), call * (unsigned)((NR - 1) * 32), tmo, call);
        __builtin_amdgcn_fence(__ATOMIC_ACQUIRE, "");
        if (__hip_atomic_load(tmo, __ATOMIC_RELAXED, __HIP_MEMORY_SCOPE_AGENT) == call)
            ((bf16_t*)a.out[rank])[(size_t)rank * NS] = 0x7fc0;
    }
}

__global__ __launch_bounds__(512, 2) void fused_kernel(Args a, int phase) {
    __shared__ __attribute__((aligned(16))) unsigned char lds[STAGE_BYTES];
    if (phase == 1) gemm_phase(a, (LAS unsigned char*)lds, lds);
    else finish_phase(a, lds);
}

}

extern "C" void kernel_launch(int rank, int nranks, ncclComm_t comm,
                              hipStream_t stream,
                              void* const* d_in, const int* in_sizes, int n_in,
                              void* d_out, int out_size,
                              const int* in_dtypes, int out_dtype,
                              void* const* d_out_all,
                              void* const* d_ws_all, size_t ws_bytes) {
    if (nranks != NR || ws_bytes < WS_NEEDED || (unsigned)rank >= (unsigned)NR) { fprintf(stderr, "unexpected configuration\n"); return; }
    Args a;
    a.x = (const bf16_t*)d_in[0]; a.w = (const bf16_t*)d_in[1]; a.rank = rank;
    a.ws = (char* const*)d_ws_all; a.out = (char* const*)d_out_all;
    fused_kernel<<<NWG, 512, 0, stream>>>(a, 1);
    fused_kernel<<<1, 512, 0, stream>>>(a, 2);
    HIP_CHECK(hipGetLastError());
}
```

```cpp
#include <hip/hip_runtime.h>
#include <rccl/rccl.h>
#include <cstdio>

#define HIP_CHECK(expr) do { hipError_t _e = (expr); if (_e != hipSuccess) { \
    fprintf(stderr, "HIP error %s at %s:%d\n", hipGetErrorString(_e), __FILE__, __LINE__); \
    return; } } while (0)

namespace {

typedef unsigned short bf16_t;
typedef short bf16x8 __attribute__((ext_vector_type(8)));
typedef float f32x4 __attribute__((ext_vector_type(4)));
typedef unsigned u32x4 __attribute__((ext_vector_type(4)));
typedef unsigned u32x2 __attribute__((ext_vector_type(2)));
#define LAS __attribute__((address_space(3)))
typedef short v4i16_t __attribute__((ext_vector_type(4)));

constexpr int NR = 8;
constexpr int GM = 4096;
constexpr int KC = 512;
constexpr int GN = 32768;
constexpr int NS = 4096;
constexpr int MS = 512;

constexpr size_t CHUNK_BYTES = (size_t)GM * KC * 2;
constexpr size_t XG_OFF = 0;
constexpr size_t WT_OFF = XG_OFF + NR * CHUNK_BYTES;
constexpr size_t FL_OFF = WT_OFF + NR * CHUNK_BYTES;
constexpr size_t IN_FLAG = FL_OFF;
constexpr size_t READY_FLAG = FL_OFF + 16384;
constexpr size_t OUT_FLAG = FL_OFF + 20480;
constexpr size_t TICKET1 = FL_OFF + 20480 + 512;
constexpr size_t TICKET2 = FL_OFF + 20480 + 768;
constexpr size_t TMO_OFF = FL_OFF + 20480 + 1024;
constexpr size_t WS_NEEDED = FL_OFF + 32768;
constexpr unsigned long long DEADLINE_TICKS = 100000000ull;

constexpr int BM = 256, BK = 64, HALF = 128, HTB = HALF * BK * 2, STAGE_BYTES = 8 * HTB, NXCD = 8, WGM = 8;
constexpr int NWG = 256;
#define NPRE0 6
#define NPRE1 18

#define STAMP(ph, i) do {} while (0)
struct Args {
    const bf16_t* x; const bf16_t* w;
    char* const* ws; char* const* out;
    int rank;
};

__device__ __forceinline__ int lds_byte(int r, int c) { const int st = (r >> 4) * 2 + (c >> 5), rr = r & 15, cc = c & 31, ob = rr * 64 + cc * 2; return st * 1024 + (ob ^ (((ob >> 9) & 1) << 5)); }
__device__ __forceinline__ void stage_rc(int b, int& R, int& C) { const int st = b / 1024, sb = b % 1024, swz = sb ^ (((sb >> 9) & 1) << 5); R = (st >> 1) * 16 + swz / 64; C = (st & 1) * 32 + (swz % 64) / 2; }
__device__ __forceinline__ int perm32(int rho) { const int n = rho >> 4, i = rho & 15; return 8 * (i >> 2) + 4 * n + (i & 3); }
__device__ __forceinline__ unsigned cvt_pk_bf16(float lo, float hi) { unsigned r; asm volatile("v_cvt_pk_bf16_f32 %0, %1, %2" : "=v"(r) : "v"(lo), "v"(hi)); return r; }

__device__ __forceinline__ unsigned long long uni64(const void* p) {
    const unsigned long long v = (unsigned long long)p;
    const unsigned lo = __builtin_amdgcn_readfirstlane((unsigned)v), hi = __builtin_amdgcn_readfirstlane((unsigned)(v >> 32));
    return ((unsigned long long)hi << 32) | lo;
}
__device__ __forceinline__ void store_wt(char* p, u32x4 v) { asm volatile("global_store_dwordx4 %0, %1, off sc0 sc1\n\ts_nop 1" :: "v"(p), "v"(v) : "memory"); }
__device__ __forceinline__ void wait_flag(const unsigned* flag, unsigned target, unsigned* tmo, unsigned call) {
    const unsigned long long t0 = __builtin_amdgcn_s_memrealtime();
    while (__hip_atomic_load(flag, __ATOMIC_RELAXED, __HIP_MEMORY_SCOPE_SYSTEM) < target) {
        if (__hip_atomic_load(tmo, __ATOMIC_RELAXED, __HIP_MEMORY_SCOPE_AGENT) == call ||
            __builtin_amdgcn_s_memrealtime() - t0 > DEADLINE_TICKS) {
            __hip_atomic_store(tmo, call, __ATOMIC_RELAXED, __HIP_MEMORY_SCOPE_AGENT);
            break;
        }
        __builtin_amdgcn_s_sleep(2);
    }
}

__device__ __forceinline__ unsigned draw_call(char* myws, size_t ticket_off, unsigned* lds_word) {
    if (threadIdx.x == 0) {
        const unsigned t = __hip_atomic_fetch_add((unsigned*)(myws + ticket_off), 1u, __ATOMIC_RELAXED, __HIP_MEMORY_SCOPE_AGENT);
        *lds_word = t / gridDim.x + 1u;
    }
    __syncthreads();
    const unsigned c = *lds_word;
    __syncthreads();
    return c;
}

__device__ void gemm_phase(const Args& a, LAS unsigned char* lds, unsigned char* lds_generic) {
    const int tid = threadIdx.x, wid = __builtin_amdgcn_readfirstlane(tid >> 6), lane = tid & 63, wr = wid >> 2, wc = wid & 3, fr = lane & 15, fq = lane >> 4;
    const int rank = a.rank;
    char* myws = a.ws[rank];
    unsigned* tmo = (unsigned*)(myws + TMO_OFF);
    STAMP(1, 0);
    const unsigned call = draw_call(myws, TICKET1, (unsigned*)lds_generic);
    constexpr int K = KC;
    const int bid = blockIdx.x;
    if (tid < NR && tid != rank) wait_flag((const unsigned*)(myws + READY_FLAG + tid * 128), call - 1u, tmo, call);
    __syncthreads();
    int pm, pn;
    { const int nM = GM / BM, nN = NS / BM, nwg = nM * nN; int wgid = blockIdx.x;
      const int q = nwg / NXCD, xcd = wgid % NXCD, off = wgid / NXCD; wgid = xcd * q + off;
      const int nig = WGM * nN, gid = wgid / nig, fm = gid * WGM; pm = fm + ((wgid % nig) % WGM); pn = (wgid % nig) / WGM; }
    const int grp = pn >> 3;
    const unsigned vx = (unsigned)(lane >> 4) * 1024u + (unsigned)(lane & 15) * 16u, vw = (unsigned)lane * 16u;
    auto seg_end = [](int g) -> int { return g < 8 ? (int)((0xA88C7E7054462A1CULL >> (8 * g)) & 255ull) : (int)((0xE0D2C4B6u >> (8 * (g - 8))) & 255u); };
    auto push_addr = [&](int step, const char*& sp, char*& dp, unsigned& voff) {
        const int i = 8 * step + wid;
        int g = 0; while (i >= seg_end(g)) ++g;
        const int idx = i - (g ? seg_end(g - 1) : 0);
        const int ks = (int)((0xBA9738625140ULL >> (4 * g)) & 15ull), kind = ks >> 2, sub = ks & 3;
        if (kind == 0) { const int gp = 28 * bid + idx, p = __builtin_amdgcn_readfirstlane((rank + 1 + (gp >> 10)) & 7);
            const size_t o = (size_t)(4 * (gp & 1023)) * 1024 + sub * 256;
            sp = (const char*)a.x + o; dp = a.ws[p] + XG_OFF + (size_t)rank * CHUNK_BYTES + o; voff = vx; }
        else { const int gp = 14 * bid + idx, p = __builtin_amdgcn_readfirstlane((rank + 1 + (gp >> 9)) & 7), r = gp & 511;
            const int row = 128 * sub + (r >> 2), cb = ((r & 3) + (kind == 2 ? 4 : 0)) * 1024;
            sp = (const char*)a.w + (size_t)row * (GN * 2) + (size_t)p * (NS * 2) + cb; dp = a.ws[p] + WT_OFF + (size_t)rank * CHUNK_BYTES + (size_t)row * (NS * 2) + cb; voff = vw; }
    };
    auto stage_steps = [](int st) -> int { return (int)((0x1C1B191712100B06ULL >> (8 * st)) & 255ull); };
    int nsig = 0;
    auto signal_upto = [&](int done_steps) {
        const int d0 = (28 * bid) >> 10, d1 = (28 * bid + 27) >> 10;
        while (nsig < 8 && stage_steps(nsig) <= done_steps) {
            if (lane == 0 || (lane == 1 && d1 != d0))
                __hip_atomic_fetch_add((unsigned*)(a.ws[(rank + 1 + (lane ? d1 : d0)) & 7] + IN_FLAG + (rank * 8 + nsig) * 128), 1u, __ATOMIC_RELAXED, __HIP_MEMORY_SCOPE_SYSTEM);
            ++nsig;
        }
    };
    const int npre = grp == 0 ? NPRE0 : NPRE1;
    for (int pb = 0; pb < npre; pb += 7) {
        u32x4 v[7];
#pragma unroll
        for (int j = 0; j < 7; ++j) if (pb + j < npre) { const char* sp; char* dp; unsigned vo; push_addr(pb + j, sp, dp, vo); v[j] = *(const u32x4*)(sp + vo); }
#pragma unroll
        for (int j = 0; j < 7; ++j) if (pb + j < npre) { const char* sp; char* dp; unsigned vo; push_addr(pb + j, sp, dp, vo); store_wt(dp + vo, v[j]); }
        asm volatile("s_waitcnt vmcnt(0)" ::: "memory");
        __syncthreads();
        if (wid == 0) signal_upto(pb + 7 < npre ? pb + 7 : npre);
    }
    u32x4 pA = {0u, 0u, 0u, 0u};
    unsigned long long arrived = 0ull;
    const unsigned* flagp; unsigned expect;
    { const int fj = lane % 7, fk = (lane / 7) & 1, fs = (lane / 14) & 3;
      const int st = (fk == 0 || grp == 0) ? (fs == 0 ? 0 : fs == 1 ? 1 : fs == 2 ? 2 : 4) : (fs == 0 ? 3 : fs == 1 ? 5 : fs == 2 ? 6 : 7);
      flagp = (const unsigned*)(myws + IN_FLAG + (((rank + 1 + fj) & 7) * 8 + st) * 128);
      const int d = 6 - fj;
      expect = call * (unsigned)((1024 * d + 1023) / 28 - (1024 * d) / 28 + 1); }
    unsigned voffA[2], krowB[2], colB[2];
#pragma unroll
    for (int i = 0; i < 2; ++i) { int R, C; stage_rc(tid * 16 + i * 8192, R, C); voffA[i] = (unsigned)(R * K + C) * 2u;
        const int k = wid * 4 + i * 32 + (lane >> 4), gi = lane & 15, f = (k & 3) | (((k >> 3) & 1) << 2);
        krowB[i] = (unsigned)k; colB[i] = (unsigned)((((gi >> 1) ^ f) << 1) | (gi & 1)) * 16u; }
    const size_t kstep = (size_t)(BK * 2), hstep = (size_t)HALF * K * 2, tstep = 2 * hstep;
    const unsigned ldsw = (unsigned)wid * 1024u;
    const int aoff = lds_byte(wr * 64 + fr, fq * 8);
    const int boff = (8 * fq + (fr >> 2)) * 256 + (((wc * 2 + ((fr & 3) >> 1)) ^ ((fr >> 2) | ((fq & 1) << 2))) * 32) + (fr & 1) * 16;
    const unsigned bbase = (unsigned)(size_t)(lds + 4 * HTB + boff);
#define SA(b, h) (((b) * 2 + (h)) * HTB)
#define SB(b, h) ((4 + (b) * 2 + (h)) * HTB)
#define STAGE(bufoff, gbase, voff) do { _Pragma("unroll") for (int _i = 0; _i < 2; ++_i) \
        __builtin_amdgcn_global_load_lds((const unsigned*)((const char*)(gbase) + (voff)[_i]), (LAS unsigned*)(lds + (bufoff) + ldsw + _i * 8192), 16, 0, 0); } while (0)
#define LDA(dst, b, h) do { _Pragma("unroll") for (int m = 0; m < 4; ++m) _Pragma("unroll") for (int k = 0; k < 2; ++k) dst[m][k] = *(const LAS bf16x8*)(lds + SA(b, h) + aoff + m * 2048 + k * 1024); } while (0)
#define TRRD(dst, off) asm volatile("ds_read_b64_tr_b16 %0, %1 offset:%2" : "=&v"(dst) : "v"(bbase), "i"(off) : "memory")
#define LDB(dst, b, h) do { _Pragma("unroll") for (int n = 0; n < 2; ++n) _Pragma("unroll") for (int k = 0; k < 2; ++k) { v4i16_t lo_, hi_; \
        TRRD(lo_, ((b) * 2 + (h)) * HTB + k * 8192 + n * 8); TRRD(hi_, ((b) * 2 + (h)) * HTB + k * 8192 + 1024 + n * 8); \
        dst[n][k] = __builtin_shufflevector(lo_, hi_, 0, 1, 2, 3, 4, 5, 6, 7); } } while (0)
#define MMA(ai, bj, At, Bt) do { __builtin_amdgcn_s_setprio(1); _Pragma("unroll") for (int m = 0; m < 4; ++m) _Pragma("unroll") for (int n = 0; n < 2; ++n) _Pragma("unroll") for (int k = 0; k < 2; ++k) \
        acc[ai][bj][m][n] = __builtin_amdgcn_mfma_f32_16x16x32_bf16(Bt[n][k], At[m][k], acc[ai][bj][m][n], 0, 0, 0); __builtin_amdgcn_s_setprio(0); } while (0)
#define WAIT_V(n) asm volatile("s_waitcnt vmcnt(" #n ")" ::: "memory")
#define WAIT_L(n) asm volatile("s_waitcnt lgkmcnt(" #n ")" ::: "memory")
#define BAR __builtin_amdgcn_s_barrier()
#define SCHED __builtin_amdgcn_sched_barrier(0)
#define PUSH_S(t) ((t) >= 1 && (t) < 29 - npre)
#define PUSH_L(t) ((t) < 28 - npre)
#define PUSH(reg, t) do { \
        if (PUSH_S(t)) { const char* sp_; char* dp_; unsigned vo_; push_addr((t) + npre - 1, sp_, dp_, vo_); \
            asm volatile("s_nop 4\n\tglobal_store_dwordx4 %1, %0, %2 sc0 sc1\n\ts_nop 1" : "+v"(reg) : "v"(vo_), "s"(uni64(dp_)) : "memory"); } \
        if (PUSH_L(t)) { const char* sp_; char* dp_; unsigned vo_; push_addr((t) + npre, sp_, dp_, vo_); \
            asm volatile("s_nop 4\n\tglobal_load_dwordx4 %0, %1, %2" : "+v"(reg) : "v"(vo_), "s"(uni64(sp_)) : "memory"); } } while (0)
#define PUSH_WAIT(t) do { if (PUSH_S(t) && PUSH_L(t)) WAIT_V(8); else if (PUSH_S(t) || PUSH_L(t)) WAIT_V(7); else WAIT_V(6); } while (0)
    f32x4 acc[2][2][4][2];
#pragma unroll
    for (int i = 0; i < 2; ++i)
#pragma unroll
        for (int j = 0; j < 2; ++j)
#pragma unroll
            for (int m = 0; m < 4; ++m)
#pragma unroll
                for (int n = 0; n < 2; ++n) acc[i][j][m][n] = (f32x4){0.f, 0.f, 0.f, 0.f};
    bf16x8 At[4][2], B0[2][2], B1[2][2];
    constexpr int NU = 32;
    auto unit_ptrs = [&](int u, const char*& pa, const char*& pb, unsigned& strideB, int& src, int& sub) {
        if (u < 4) { src = rank; sub = u;
            pa = (const char*)a.x + (size_t)pm * tstep + sub * 256;
            pb = (const char*)a.w + (size_t)(sub * 128) * (GN * 2) + (size_t)(rank * NS + pn * BM) * 2; strideB = GN * 2; }
        else { const int v = u - 4; sub = v / 7; src = (rank + 1 + v % 7) & 7;
            pa = myws + XG_OFF + (size_t)src * CHUNK_BYTES + (size_t)pm * tstep + sub * 256;
            pb = myws + WT_OFF + (size_t)src * CHUNK_BYTES + (size_t)(sub * 128) * (NS * 2) + pn * (BM * 2); strideB = NS * 2; }
    };
    const char *cA, *nB; unsigned sB; int src_, sub_;
    unsigned voffB[2];
    unit_ptrs(0, cA, nB, sB, src_, sub_);
    voffB[0] = krowB[0] * sB + colB[0]; voffB[1] = krowB[1] * sB + colB[1];
    STAGE(SB(0, 0), nB, voffB); STAGE(SA(0, 0), cA, voffA); STAGE(SB(0, 1), nB + 256, voffB); STAGE(SA(0, 1), cA + hstep, voffA);
    if (wr == 1) BAR;
    WAIT_V(4); BAR;
    STAGE(SB(1, 0), nB + (size_t)64 * sB, voffB); STAGE(SA(1, 0), cA + kstep, voffA); STAGE(SB(1, 1), nB + (size_t)64 * sB + 256, voffB);
    WAIT_V(6); BAR;
    for (int u = 0; u < NU; ++u) {
        const bool has_next = u + 1 < NU;
        const char* nA;
        unit_ptrs(has_next ? u + 1 : u, nA, nB, sB, src_, sub_);
        voffB[0] = krowB[0] * sB + colB[0]; voffB[1] = krowB[1] * sB + colB[1];
        const char* a1 = cA + kstep;
        const char* a2 = nA; const char* b2 = nB;
        const char* a3 = a2 + kstep; const char* b3 = b2 + (size_t)64 * sB;
        if (wid == 0) {
            if (u >= 3) signal_upto(npre + u - 2);
            const unsigned long long need = 0x81ull << (14 * ((u - 3) / 7) + (u - 3) % 7);
            if (u >= 3 && has_next && (arrived & need) != need) {
                const unsigned long long t0 = __builtin_amdgcn_s_memrealtime();
                for (;;) {
                    const unsigned v = lane < 56 ? __hip_atomic_load(flagp, __ATOMIC_RELAXED, __HIP_MEMORY_SCOPE_SYSTEM) : 0u;
                    arrived = __builtin_amdgcn_ballot_w64(v >= expect);
                    if ((arrived & need) == need) break;
                    if (__hip_atomic_load(tmo, __ATOMIC_RELAXED, __HIP_MEMORY_SCOPE_AGENT) == call || __builtin_amdgcn_s_memrealtime() - t0 > DEADLINE_TICKS) {
                        __hip_atomic_store(tmo, call, __ATOMIC_RELAXED, __HIP_MEMORY_SCOPE_AGENT); arrived = ~0ull; break; }
                    __builtin_amdgcn_s_sleep(1);
                }
                __builtin_amdgcn_fence(__ATOMIC_ACQUIRE, "");
                asm volatile("s_waitcnt vmcnt(0)" ::: "memory");
            }
        }
        asm volatile("" ::: "memory"); BAR; asm volatile("" ::: "memory");
        LDB(B0, 0, 0); SCHED; LDA(At, 0, 0); STAGE(SA(1, 1), a1 + hstep, voffA);
        WAIT_L(8); BAR; WAIT_L(0); SCHED; MMA(0, 0, At, B0); BAR; SCHED;
        LDB(B1, 0, 1); STAGE(SB(0, 0), b2, voffB);
        BAR; WAIT_L(0); SCHED; MMA(0, 1, At, B1); BAR;
        LDA(At, 0, 1); STAGE(SA(0, 0), a2, voffA);
        BAR; WAIT_L(0); SCHED; MMA(1, 0, At, B0); BAR; SCHED;
        STAGE(SB(0, 1), b2 + 256, voffB);
        WAIT_V(6);
        BAR; MMA(1, 1, At, B1); BAR;
        LDB(B0, 1, 0); SCHED; LDA(At, 1, 0); STAGE(SA(0, 1), a2 + hstep, voffA);
        PUSH(pA, u);
        WAIT_L(8); BAR; WAIT_L(0); SCHED; MMA(0, 0, At, B0); BAR; SCHED;
        LDB(B1, 1, 1); STAGE(SB(1, 0), b3, voffB);
        BAR; WAIT_L(0); SCHED; MMA(0, 1, At, B1); BAR;
        LDA(At, 1, 1); STAGE(SA(1, 0), a3, voffA);
        BAR; WAIT_L(0); SCHED; MMA(1, 0, At, B0); BAR; SCHED;
        STAGE(SB(1, 1), b3 + 256, voffB);
        PUSH_WAIT(u);
        BAR; MMA(1, 1, At, B1); BAR;
        cA = nA;
        if ((u & 3) == 3) STAMP(1, 2 + (u >> 2));
    }
    WAIT_V(0);
    if (wr == 0) BAR;
    BAR;
    const bool bad = __hip_atomic_load(tmo, __ATOMIC_RELAXED, __HIP_MEMORY_SCOPE_AGENT) == call;
    const int owner = pm >> 1;
    {
        bf16_t* obase = (bf16_t*)a.out[owner] + (size_t)((pm & 1) * BM + wr * 64 + fr) * GN + (size_t)rank * NS + pn * BM + wc * 32 + 8 * fq;
#pragma unroll
        for (int ai = 0; ai < 2; ++ai)
#pragma unroll
            for (int m = 0; m < 4; ++m) {
                bf16_t* rowp = obase + (size_t)(ai * HALF + m * 16) * GN;
#pragma unroll
                for (int bj = 0; bj < 2; ++bj) {
                    const f32x4 v0 = acc[ai][bj][m][0], v1 = acc[ai][bj][m][1];
                    u32x4 w;
                    w.x = cvt_pk_bf16(fmaxf(v0[0], 0.f), fmaxf(v0[1], 0.f)); w.y = cvt_pk_bf16(fmaxf(v0[2], 0.f), fmaxf(v0[3], 0.f));
                    w.z = cvt_pk_bf16(fmaxf(v1[0], 0.f), fmaxf(v1[1], 0.f)); w.w = cvt_pk_bf16(fmaxf(v1[2], 0.f), fmaxf(v1[3], 0.f));
                    if (bad) { w.x = 0x7fc07fc0u; w.y = 0x7fc07fc0u; w.z = 0x7fc07fc0u; w.w = 0x7fc07fc0u; }
                    store_wt((char*)(rowp + bj * HALF), w);
                }
            }
    }
    STAMP(1, 10);
    asm volatile("s_waitcnt vmcnt(0)" ::: "memory");
    __syncthreads();
    STAMP(1, 11);
    if (wid == 0) {
        if (lane == 0 && owner != rank)
            __hip_atomic_fetch_add((unsigned*)(a.ws[owner] + OUT_FLAG), 1u, __ATOMIC_RELAXED, __HIP_MEMORY_SCOPE_SYSTEM);
        unsigned t = 0;
        if (lane == 0) t = __hip_atomic_fetch_add((unsigned*)(myws + TICKET2), 1u, __ATOMIC_RELAXED, __HIP_MEMORY_SCOPE_AGENT);
        t = __builtin_amdgcn_readfirstlane(t);
        if (t == call * (unsigned)NWG - 1u) {
            if (lane < NR && lane != rank)
                __hip_atomic_store((unsigned*)(a.ws[lane] + READY_FLAG + rank * 128), call, __ATOMIC_RELAXED, __HIP_MEMORY_SCOPE_SYSTEM);
            if (lane == 0) {
                wait_flag((const unsigned*)(myws + OUT_FLAG), call * (unsigned)((NR - 1) * 32), tmo, call);
                __builtin_amdgcn_fence(__ATOMIC_ACQUIRE, "");
                if (__hip_atomic_load(tmo, __ATOMIC_RELAXED, __HIP_MEMORY_SCOPE_AGENT) == call)
                    ((bf16_t*)a.out[rank])[(size_t)rank * NS] = 0x7fc0;
            }
        }
    }
    STAMP(1, 12);
#undef SA
#undef SB
#undef STAGE
#undef LDA
#undef LDB
#undef MMA
}

__global__ __launch_bounds__(512, 2) void fused_kernel(Args a, int phase) {
    __shared__ __attribute__((aligned(16))) unsigned char lds[STAGE_BYTES];
    gemm_phase(a, (LAS unsigned char*)lds, lds);
}

}

extern "C" void kernel_launch(int rank, int nranks, ncclComm_t comm,
                              hipStream_t stream,
                              void* const* d_in, const int* in_sizes, int n_in,
                              void* d_out, int out_size,
                              const int* in_dtypes, int out_dtype,
                              void* const* d_out_all,
                              void* const* d_ws_all, size_t ws_bytes) {
    if (nranks != NR || ws_bytes < WS_NEEDED || (unsigned)rank >= (unsigned)NR) { fprintf(stderr, "unexpected configuration\n"); return; }
    Args a;
    a.x = (const bf16_t*)d_in[0]; a.w = (const bf16_t*)d_in[1]; a.rank = rank;
    a.ws = (char* const*)d_ws_all; a.out = (char* const*)d_out_all;
    fused_kernel<<<NWG, 512, 0, stream>>>(a, 1);
    HIP_CHECK(hipGetLastError());
}
```
